```python
import numpy as np

kernel_name = "gpu_gemm_rs_m4096_k4096_n16384_bf16_none"


ABSMAX_RTOL = 0.008

COMPUTE_DTYPE = "bfloat16"
EPILOGUE = "none"
IN_QUANT = None
OUT_QUANT = None
IN_DTYPES = {"x": "bfloat16", "w_mat": "bfloat16"}
WORLD_SIZE = 8

def _to_bf16(x):
    u = np.ascontiguousarray(x, dtype=np.float32).view(np.uint32)
    return ((u + 0x7FFF + ((u >> 16) & 1)) & 0xFFFF0000).view(np.float32)

def setup_inputs():

    x = _to_bf16(np.random.randn(4096, 4096).astype(np.float32))
    w_mat = _to_bf16((np.random.randn(4096, 16384) / np.sqrt(4096)).astype(np.float32))
    return {"x": x, "w_mat": w_mat}

def _epilogue(y):
    return y

def reference(x, w_mat):
    y = (x @ w_mat).astype(np.float32)
    return _epilogue(y)
```

```cpp
#include <hip/hip_runtime.h>
#include <rccl/rccl.h>
#include <cstdio>
#include <cstdint>

#define HIP_CHECK(expr) do { hipError_t _e = (expr); if (_e != hipSuccess) { \
    fprintf(stderr, "HIP error %s at %s:%d\n", hipGetErrorString(_e), __FILE__, __LINE__); \
    return; } } while (0)

namespace {

typedef unsigned u32x4 __attribute__((ext_vector_type(4)));
typedef unsigned u32x2 __attribute__((ext_vector_type(2)));
typedef float f32x4 __attribute__((ext_vector_type(4)));
typedef __bf16 bf16x8 __attribute__((ext_vector_type(8)));
typedef __bf16 bf16x2 __attribute__((ext_vector_type(2)));

constexpr int NR = 8;
constexpr int MTOT = 4096, KTOT = 4096, NTOT = 16384;
constexpr int KSH = KTOT / NR;
constexpr int NSH = NTOT / NR;
constexpr int MSH = MTOT / NR;
constexpr int NWG = 256;
constexpr int NTHREADS = 768;

constexpr unsigned OFF_CTR   = 0;
constexpr unsigned OFF_TMO   = 32768;
constexpr unsigned OFF_TICK  = 33792;
constexpr unsigned OFF_ACT   = 35840;
constexpr unsigned OFF_WFLAG = 49152;
constexpr unsigned OFF_BCNT  = 65536;
constexpr unsigned OFF_FLAGS = 524288;
constexpr size_t   OFF_STAGE = 2u << 20;
constexpr unsigned STAGE_BYTES = 48u << 20;
constexpr unsigned STAGE_W = 32u << 20;
constexpr size_t   OFF_WP  = OFF_STAGE + 2 * (size_t)STAGE_BYTES;
constexpr unsigned WP_BYTES = 16u << 20;
constexpr size_t   OFF_YP  = OFF_WP + WP_BYTES;
constexpr unsigned YP_BYTES = 16u << 20;
constexpr size_t   WS_NEEDED = OFF_YP + YP_BYTES;
constexpr unsigned OUT_BYTES = (unsigned)MSH * NTOT * 2;

constexpr unsigned SLOT_BYTES = 49152;
constexpr unsigned LDS_YT   = 49152;
constexpr unsigned DSLOT_BYTES = 24576;
constexpr unsigned LDS_D0 = 114688, LDS_D1 = 0, LDS_D2 = 24576;
constexpr unsigned LDS_CTL  = 3 * SLOT_BYTES;
constexpr unsigned C_READY = 0, C_PAR = 4, C_ABORT = 8, C_FULL = 16, C_FREE = 32, C_YD = 64, C_ORDER = 128;
constexpr unsigned LDS_TOTAL = LDS_CTL + 512;

#define WSH 1
#define XSH 1
#define NSPLIT 4
#define POLL_SLEEP 32
#define WINPAT 0x22222222u
#define WIN_Y 3
#define YPROP 1
#define YWT 72
#define YWCAP 8
#define YW0 4
#define YTRIG 2
#define YSLEEP 4
#define OROT 1
#define EARLYSND 1
#define WPN0 2
#define WPNS 3
constexpr unsigned NUNITS = 704;
constexpr unsigned TICKS_PER_CALL = NUNITS - 32 + 64;

constexpr unsigned long long DEADLINE = 100000000ull;

#define PT_DECL
#define PT(i)
#define PT_FLUSH(lo, hi)

struct Params {
    const char* x;
    const char* w;
    char* out[NR];
    char* ws[NR];
    int rank;
};

#define VMCNT(n) asm volatile("s_waitcnt vmcnt(" #n ")" ::: "memory")
#define LGKM0()  asm volatile("s_waitcnt lgkmcnt(0)" ::: "memory")

__device__ __forceinline__ unsigned long long rt() { return __builtin_amdgcn_s_memrealtime(); }

__device__ __forceinline__ unsigned lds_ld(unsigned addr) {
    unsigned v;
    asm volatile("ds_read_b32 %0, %1\n\ts_waitcnt lgkmcnt(0)" : "=&v"(v) : "v"(addr) : "memory");
    return v;
}
__device__ __forceinline__ void lds_st(unsigned addr, unsigned v) {
    asm volatile("ds_write_b32 %0, %1" : : "v"(addr), "v"(v) : "memory");
}
__device__ __forceinline__ void glds4(unsigned lane16, unsigned l, const char* g) {
    unsigned keep;
    asm volatile("s_nop 4\n\ts_mov_b32 %0, m0\n\ts_mov_b32 m0, %2\n\ts_nop 0\n\t"
                 "global_load_lds_dwordx4 %1, %3 sc0 sc1\n\t"
                 "global_load_lds_dwordx4 %1, %3 offset:1024 sc0 sc1\n\t"
                 "global_load_lds_dwordx4 %1, %3 offset:2048 sc0 sc1\n\t"
                 "global_load_lds_dwordx4 %1, %3 offset:3072 sc0 sc1\n\t"
                 "s_mov_b32 m0, %0"
                 : "=&s"(keep) : "v"(lane16), "s"(l), "s"(g) : "memory");
}
__device__ __forceinline__ void glds4g(unsigned v0, unsigned v1, unsigned v2, unsigned v3, unsigned l, const char* g) {
    unsigned keep;
    asm volatile("s_nop 4\n\ts_mov_b32 %0, m0\n\ts_mov_b32 m0, %5\n\ts_nop 0\n\t"
                 "global_load_lds_dwordx4 %1, %6\n\t"
                 "global_load_lds_dwordx4 %2, %6 offset:1024\n\t"
                 "global_load_lds_dwordx4 %3, %6 offset:2048\n\t"
                 "global_load_lds_dwordx4 %4, %6 offset:3072\n\t"
                 "s_mov_b32 m0, %0"
                 : "=&s"(keep) : "v"(v0), "v"(v1), "v"(v2), "v"(v3), "s"(l), "s"(g) : "memory");
}

__device__ __forceinline__ void lds_wait_all(unsigned ctl, unsigned off, int n, unsigned want, int lane) {
    const unsigned a = ctl + off + (unsigned)(lane < n ? lane : 0) * 4u;
    if (__all(lds_ld(a) >= want)) return;
    const unsigned long long t0 = rt();
    for (unsigned sp = 0;;) {
        __builtin_amdgcn_s_sleep(1);
        if (__all(lds_ld(a) >= want)) return;
        if ((++sp & 1023u) == 0u && rt() - t0 > 2 * DEADLINE) { lds_st(ctl + C_ABORT, 1u); return; }
    }
}

constexpr int win_of(int k) { return (int)((WINPAT >> (4 * k)) & 15u); }
constexpr int flag_pos() {
    for (int k = 0; k < 8; ++k) if (k - win_of(k) >= -1) return k;
    return 8;
}
constexpr int FLAGPOS = flag_pos();
constexpr int cnt_pos() {
    for (int k = FLAGPOS + 1; k < 8; ++k) if (k - win_of(k) >= FLAGPOS) return k;
    return 8;
}
constexpr int CPOS = cnt_pos();
static_assert(CPOS < 4, "window too large");
constexpr int snd_allowed(int k) {
    int pos[8] = {0, 0, 0, 0, 0, 0, 0, 0}, pre[8] = {0, 0, 0, 0, 0, 0, 0, 0};
    int n = 0;
    for (int i = 0; i < 8; ++i) {
        pre[i] = n;
        if (i == FLAGPOS) ++n;
        if (i == CPOS) ++n;
        pos[i] = n++;
        if (i == 0) ++n;
        if (i < 4) n += 2;
    }
    const int w = win_of(k);
    const int a = k >= w ? pre[k] - pos[k - w] - 1 : pre[k] + n - pos[8 + k - w] - 1;
    return (k == 0 && a > 4) ? 4 : a;
}
#define SWAIT(k) asm volatile("s_waitcnt vmcnt(%0)" :: "n"(snd_allowed(k)) : "memory")

__device__ __forceinline__ void unit_of(unsigned u, int& type, int& c, int& j) {
    unsigned v;
    if (u < 88u * NSPLIT) { c = (int)(u / 88u); v = u - (unsigned)c * 88u; }
    else {
        u -= 88u * NSPLIT;
        if (u < 60u * (8 - NSPLIT)) { const unsigned s = u / 60u; c = NSPLIT + (int)s; v = u - s * 60u; }
        else { u -= 60u * (8 - NSPLIT); const unsigned s = u / 28u; c = NSPLIT + (int)s; v = 60u + u - s * 28u; }
    }
    type = v < 32u ? 0 : (v < 60u ? 1 : 2);
    j = (int)v - (type == 0 ? 0 : (type == 1 ? 32 : 60));
}

#define TRANSPOSE8(T, R) do { _Pragma("unroll") for (int jj = 0; jj < 8; ++jj) _Pragma("unroll") for (int t = 0; t < 4; ++t) \
        T[jj][t] = __builtin_amdgcn_perm(R[2 * t + 1][jj >> 1], R[2 * t][jj >> 1], (jj & 1) ? 0x07060302u : 0x05040100u); } while (0)

__global__ void __launch_bounds__(NTHREADS) fused_gemm_rs(Params P) {
    extern __shared__ __attribute__((aligned(16))) char smem[];
    const int tid = threadIdx.x;
    const int lane = tid & 63;
    const int wave = __builtin_amdgcn_readfirstlane(tid >> 6);
    const int b = blockIdx.x;
    const int rank = P.rank;
    const int xcd = b & 7, idx = b >> 3;
    const unsigned q = (unsigned)(rank + xcd + idx) & 7u;
    const int ml = ((xcd >> 1) + (int)q) & 3;
    const int mt = (int)q * 4 + ml;
    const int nt = (xcd & 1) * 4 + (idx >> 3);
    const int mlp = idx >> 3, ntp = xcd;
    const unsigned ldsb = (unsigned)(size_t)smem;
    const unsigned ctl = ldsb + LDS_CTL;
    char* const myws = P.ws[rank];
    unsigned* const tmo = (unsigned*)(myws + OFF_TMO);

    if (wave == 0) { if (lane < 32) { lds_st(ctl + lane * 4, 0u); } LGKM0(); }
    if (wave != 11) __builtin_amdgcn_s_barrier();
    unsigned* const ctr = (unsigned*)(myws + OFF_CTR + b * 128);

    if (wave < 8) {
        PT_DECL;
        const int g = lane >> 4, cc = lane & 15;
        const unsigned ctrv = __hip_atomic_load(ctr, __ATOMIC_RELAXED, __HIP_MEMORY_SCOPE_AGENT);
        const int wm = wave >> 2, wn = wave & 3;
        f32x4 acc[4][4];
#pragma unroll
        for (int i = 0; i < 4; ++i)
#pragma unroll
            for (int j = 0; j < 4; ++j) acc[i][j] = (f32x4){0.f, 0.f, 0.f, 0.f};
        auto wait_full = [&](unsigned want) {
            const unsigned wa = ctl + C_FULL + (lane & 1) * 4;
            if (!__all(lds_ld(wa) >= want)) {
                const unsigned long long t0 = rt();
                for (unsigned sp = 0;;) {
                    __builtin_amdgcn_s_sleep(1);
                    if (__all(lds_ld(wa) >= want)) break;
                    if ((++sp & 1023u) == 0u && rt() - t0 > 2 * DEADLINE) { lds_st(ctl + C_ABORT, 1u); break; }
                }
            }
        };
        {
            const unsigned aoff = wm * 8192 + lane * 16;
            const unsigned boff = 16384 + (wn >> 1) * 8192 + (wn & 1) * 4096 + lane * 16;
            unsigned sb = 2 * SLOT_BYTES;
            for (int n = 0; n < 56; ++n) {
                if (n == WPN0 + WPNS * wave) {
                    const int c = mlp * 2 + (wave >> 2), kh = (wave >> 1) & 1, cgl = wave & 1;
                    const char* src = P.w + (size_t)(c * 64 + kh * 32 + g * 8) * (NTOT * 2) + (size_t)q * (NSH * 2) + ntp * 512 + cgl * 256 + cc * 16;
                    u32x4 R[8], T[8];
#pragma unroll
                    for (int t = 0; t < 8; ++t) R[t] = *(const u32x4*)(src + (size_t)t * (NTOT * 2));
                    TRANSPOSE8(T, R);
                    const auto wrs = __builtin_amdgcn_make_buffer_rsrc(myws + OFF_WP, 0, WP_BYTES, 0x00020000);
                    const unsigned off = ((q * 8 + c) * 8 + ntp) * 32768u + (kh * 2 + cgl) * 8192u + lane * 16;
#pragma unroll
                    for (int t = 0; t < 8; ++t) __builtin_amdgcn_raw_buffer_store_b128(T[t], wrs, off + t * 1024, 0, 17);
                    VMCNT(0);
                    const unsigned call = __builtin_amdgcn_readfirstlane(ctrv) + 1u;
                    if (lane == 0)
                        __hip_atomic_store((unsigned*)(myws + OFF_WFLAG) + (q * 8 + ntp) * 32 + c * 4 + kh * 2 + cgl, call, __ATOMIC_RELAXED, __HIP_MEMORY_SCOPE_AGENT);
                    PT(60);
                }
                wait_full((unsigned)n + 1u);
                if ((n & 3) == 0) PT(32 + (n >> 2));
                const char* s = smem + sb;
                bf16x8 a[2][4], bb[2][4];
#pragma unroll
                for (int kh = 0; kh < 2; ++kh) {
#pragma unroll
                    for (int i = 0; i < 4; ++i) a[kh][i] = *(const bf16x8*)(s + aoff + (i * 2 + kh) * 1024);
#pragma unroll
                    for (int j = 0; j < 4; ++j) bb[kh][j] = *(const bf16x8*)(s + boff + kh * 16384 + j * 1024);
                }
                __builtin_amdgcn_sched_barrier(0);
#pragma unroll
                for (int i = 0; i < 4; ++i)
#pragma unroll
                    for (int j = 0; j < 4; ++j)
                        acc[i][j] = __builtin_amdgcn_mfma_f32_16x16x32_bf16(a[0][i], bb[0][j], acc[i][j], 0, 0, 0);
                __builtin_amdgcn_sched_barrier(0);
                LGKM0();
                if (lane == 0) lds_st(ctl + C_FREE + wave * 4, (unsigned)n + 1u);
                __builtin_amdgcn_sched_barrier(0);
#pragma unroll
                for (int i = 0; i < 4; ++i)
#pragma unroll
                    for (int j = 0; j < 4; ++j)
                        acc[i][j] = __builtin_amdgcn_mfma_f32_16x16x32_bf16(a[1][i], bb[1][j], acc[i][j], 0, 0, 0);
                sb += SLOT_BYTES; if (sb == 3 * SLOT_BYTES) sb = 0;
            }
        }
        PT(56);
        lds_wait_all(ctl, C_FREE, 8, 55u, lane);
        {
            char* yt = smem + LDS_YT + (wm * 64 + g * 4) * 512 + ((wn >> 1) * 128 + cc * 8 + (wn & 1) * 4) * 2;
#pragma unroll
            for (int i = 0; i < 4; ++i)
#pragma unroll
                for (int r = 0; r < 4; ++r) {
                    bf16x2 lo, hi;
                    lo.x = (__bf16)acc[i][0][r]; lo.y = (__bf16)acc[i][1][r];
                    hi.x = (__bf16)acc[i][2][r]; hi.y = (__bf16)acc[i][3][r];
                    u32x2 v; v.x = __builtin_bit_cast(unsigned, lo); v.y = __builtin_bit_cast(unsigned, hi);
                    v.x = v.x ? v.x : 0x80008000u; v.y = v.y ? v.y : 0x80008000u;
                    *(u32x2*)(yt + (i * 16 + r) * 512) = v;
                }
        }
        LGKM0();
        if (lane == 0) lds_st(ctl + C_YD + wave * 4, 1u);
        PT(57);
        const int wm4 = wave >> 1, cg = wave & 1;
        f32x4 dac[2][8];
#pragma unroll
        for (int i = 0; i < 2; ++i)
#pragma unroll
            for (int j = 0; j < 8; ++j) dac[i][j] = (f32x4){0.f, 0.f, 0.f, 0.f};
        {
            const unsigned aoff = wm4 * 2048 + lane * 16;
            const unsigned boff = 8192 + cg * 8192 + lane * 16;
            unsigned sb = LDS_D0;
            for (int h = 0; h < 16; ++h) {
                wait_full(57u + (unsigned)h);
                const char* s = smem + sb;
                bf16x8 a[2], bb[8];
#pragma unroll
                for (int i = 0; i < 2; ++i) a[i] = *(const bf16x8*)(s + aoff + i * 1024);
#pragma unroll
                for (int j = 0; j < 8; ++j) bb[j] = *(const bf16x8*)(s + boff + j * 1024);
                LGKM0();
                if (lane == 0) lds_st(ctl + C_FREE + wave * 4, 57u + (unsigned)h);
                __builtin_amdgcn_sched_barrier(0);
#pragma unroll
                for (int i = 0; i < 2; ++i)
#pragma unroll
                    for (int j = 0; j < 8; ++j)
                        dac[i][j] = __builtin_amdgcn_mfma_f32_16x16x32_bf16(a[i], bb[j], dac[i][j], 0, 0, 0);
                sb = sb == LDS_D0 ? LDS_D1 : (sb == LDS_D1 ? LDS_D2 : LDS_D0);
            }
        }
        PT(61);
        {
            const unsigned call = __builtin_amdgcn_readfirstlane(ctrv) + 1u;
            const auto ors = __builtin_amdgcn_make_buffer_rsrc(P.out[rank], 0, OUT_BYTES, 0x00020000);
            const unsigned voff = (unsigned)(mlp * 128 + wm4 * 32 + g * 4) * (NTOT * 2) + q * (NSH * 2) + ntp * 512 + cg * 256 + cc * 16;
            const auto prs = __builtin_amdgcn_make_buffer_rsrc(myws + OFF_YP, 0, YP_BYTES, 0x00020000);
            const unsigned pbase = (q * 32 + mlp * 8 + ntp) * 65536u + wm4 * 16384 + cg * 1024;
            const unsigned trig = wm4 == 3 ? 0u : YTRIG * 2048u;
            const unsigned long long t0 = rt();
            bool bad = false;
            bool eager = true;
#pragma unroll
            for (int i = 0; i < 2; ++i) {
                const unsigned poff = pbase + i * 8192 + lane * 16;
                u32x4 pv[4];
                unsigned done = 0, sp = 0;
                bool first = true;
                for (;;) {
                    if (!(eager && first) && !(done >> (trig >> 11) & 1u)) {
                        const unsigned* const tp = (const unsigned*)(myws + OFF_YP + pbase + i * 8192 + trig);
                        while (__hip_atomic_load(tp, __ATOMIC_RELAXED, __HIP_MEMORY_SCOPE_SYSTEM) == 0u) {
                            __builtin_amdgcn_s_sleep(YSLEEP);
                            if ((++sp & 63u) == 0u && (lds_ld(ctl + C_ABORT) != 0u || rt() - t0 > DEADLINE
                                    || __hip_atomic_load(tmo, __ATOMIC_RELAXED, __HIP_MEMORY_SCOPE_AGENT) == call)) { bad = true; break; }
                        }
                    }
                    asm volatile("" ::: "memory");
#pragma unroll
                    for (int r = 0; r < 4; ++r)
                        if (!(done >> r & 1u)) pv[r] = __builtin_amdgcn_raw_buffer_load_b128(prs, poff + r * 2048, 0, 17);
                    const unsigned tv = __hip_atomic_load(tmo, __ATOMIC_RELAXED, __HIP_MEMORY_SCOPE_AGENT);
                    asm volatile("" ::: "memory");
#pragma unroll
                    for (int r = 0; r < 4; ++r) {
                        const bool ok = (pv[r][0] != 0u) & (pv[r][1] != 0u) & (pv[r][2] != 0u) & (pv[r][3] != 0u);
                        if (__all(ok)) done |= 1u << r;
                    }
                    bad = bad | (__builtin_amdgcn_readfirstlane(tv) == call);
                    if (done == 15u) break;
                    first = false;
                    if (bad || ((++sp & 63u) == 0u && (lds_ld(ctl + C_ABORT) != 0u || rt() - t0 > DEADLINE))) { bad = true; break; }
                }
                eager = eager && first;
                bad = bad | (lds_ld(ctl + C_ABORT) != 0u);
                if (bad && lane == 0) { __hip_atomic_store(tmo, call, __ATOMIC_RELAXED, __HIP_MEMORY_SCOPE_AGENT); lds_st(ctl + C_ABORT, 1u); }
                if (i == 1) PT(58);
#pragma unroll
                for (int r = 0; r < 4; ++r) {
                    u32x4 o;
#pragma unroll
                    for (int e = 0; e < 4; ++e) {
                        const float lo = __builtin_bit_cast(float, pv[r][e] << 16) + dac[i][2 * e][r];
                        const float hi = __builtin_bit_cast(float, pv[r][e] & 0xFFFF0000u) + dac[i][2 * e + 1][r];
                        bf16x2 pk; pk.x = (__bf16)lo; pk.y = (__bf16)hi;
                        o[e] = bad ? 0x7FC07FC0u : __builtin_bit_cast(unsigned, pk);
                    }
                    __builtin_amdgcn_raw_buffer_store_b128(o, ors, voff + (i * 16 + r) * (NTOT * 2), 0, 17);
                }
#pragma unroll
                for (int r = 0; r < 4; ++r)
                    __builtin_amdgcn_raw_buffer_store_b128((u32x4){0u, 0u, 0u, 0u}, prs, poff + r * 2048, 0, 17);
            }
        }
        VMCNT(0);
        PT(59);
        if (wave == 7) { PT_FLUSH(32, 48); PT_FLUSH(56, 62); }
        return;
    }
    if (wave < 10) {
        __builtin_amdgcn_s_setprio(2);
        const int lw = wave - 8;
        const unsigned lane16 = lane * 16;
        const unsigned fullw = ctl + C_FULL + lw * 4;
        const unsigned gv = (unsigned)(lane & 15) * 1024u + (unsigned)(lane >> 4) * 16u + 4096u;
        const char* const xd = P.x + (size_t)(rank * MSH + mlp * 128) * (KSH * 2) - 4096;
        int pub = 0;
        unsigned sb = 2 * SLOT_BYTES;
        const char* stage = nullptr;
        for (int n = 0; n < 72; ++n) {
            const unsigned pa = ctl + (lane < 8 ? C_FREE + lane * 4 : C_READY);
            const unsigned nfree = n < 56 ? (unsigned)(n > 2 ? n - 2 : 0) : (n == 56 ? 55u : (n == 57 ? 56u : (unsigned)n - 2u));
            const unsigned need = lane < 8 ? nfree : (unsigned)n + 1u;
            if (!__all(lds_ld(pa) >= need)) {
                if (pub < n) { VMCNT(0); pub = n; if (lane == 0) lds_st(fullw, (unsigned)pub); }
                const unsigned long long t0 = rt();
                for (unsigned sp = 0;;) {
                    if (__all(lds_ld(pa) >= need)) break;
                    __builtin_amdgcn_s_sleep(1);
                    if ((++sp & 1023u) == 0u && rt() - t0 > 2 * DEADLINE) { lds_st(ctl + C_ABORT, 1u); break; }
                }
            }
            if (n == 0) {
                const unsigned par = __builtin_amdgcn_readfirstlane(lds_ld(ctl + C_PAR)) & 1u;
                stage = myws + OFF_STAGE + (size_t)par * STAGE_BYTES;
            }
            const unsigned l = ldsb + sb;
            if (n < 56) {
                const unsigned ks = __builtin_amdgcn_readfirstlane(lds_ld(ctl + C_ORDER + n * 4)) & 63u;
                const char* wsrc = stage + STAGE_W + (size_t)(ks * 8 + nt) * 32768;
                if (lw == 0) {
                    const char* xs = stage + (size_t)(ks * 32 + mt) * 16384;
#pragma unroll
                    for (int i = 0; i < 4; ++i) glds4(lane16, l + i * 4096, xs + i * 4096);
#pragma unroll
                    for (int i = 0; i < 2; ++i) glds4(lane16, l + 16384 + i * 4096, wsrc + i * 4096);
                } else {
#pragma unroll
                    for (int i = 2; i < 8; ++i) glds4(lane16, l + 16384 + i * 4096, wsrc + i * 4096);
                }
                if (n + 1 - pub == 2) { VMCNT(24); ++pub; if (lane == 0) lds_st(fullw, (unsigned)pub); }
                sb += SLOT_BYTES; if (sb == 3 * SLOT_BYTES) sb = 0;
                if (n == 55) sb = LDS_D0;
            } else {
                const int h = n - 56;
                const char* wsrc = myws + OFF_WP + (size_t)((q * 8 + (h >> 1)) * 8 + ntp) * 32768 + (h & 1) * 16384;
                if (lw == 0) {
#pragma unroll
                    for (int i = 0; i < 2; ++i)
                        glds4g(gv, gv + 16384 - 1024, gv + 32768 - 2048, gv + 49152 - 3072, l + i * 4096, xd + i * 65536 + h * 64);
                    glds4(lane16, l + 8192, wsrc);
                } else {
#pragma unroll
                    for (int i = 1; i < 4; ++i) glds4(lane16, l + 8192 + i * 4096, wsrc + i * 4096);
                }
                if (n + 1 - pub == 2) { VMCNT(12); ++pub; if (lane == 0) lds_st(fullw, (unsigned)pub); }
                sb = sb == LDS_D0 ? LDS_D1 : (sb == LDS_D1 ? LDS_D2 : LDS_D0);
            }
        }
        VMCNT(0);
        if (lane == 0) lds_st(fullw, 72u);
        return;
    }
    const unsigned ctrv = __hip_atomic_load(ctr, __ATOMIC_RELAXED, __HIP_MEMORY_SCOPE_AGENT);
    if (wave == 10) {
        const unsigned call = __builtin_amdgcn_readfirstlane(ctrv) + 1u;
        __builtin_amdgcn_s_setprio(1);
        if (lane == 0) lds_st(ctl + C_PAR, call);
        bool dead = false;
        const unsigned long long t0 = rt();
        PT_DECL;
        unsigned sp = 0;
        auto idle = [&]() {
            __builtin_amdgcn_s_sleep(POLL_SLEEP);
            if ((++sp & 63u) == 0u) {
                if (__builtin_amdgcn_readfirstlane(__hip_atomic_load(tmo, __ATOMIC_RELAXED, __HIP_MEMORY_SCOPE_AGENT)) == call
                    || rt() - t0 > DEADLINE) {
                    if (lane == 0) { __hip_atomic_store(tmo, call, __ATOMIC_RELAXED, __HIP_MEMORY_SCOPE_AGENT); lds_st(ctl + C_ABORT, 1u); }
                    dead = true;
                }
            }
        };
        unsigned n = 0;
        const int pp = (lane >> 1) & 7, f = lane & 1;
        const unsigned p = (unsigned)(rank + pp) & 7u;
        const unsigned blk = f ? 8u + ((unsigned)mt >> XSH) : (unsigned)nt >> WSH;
        const unsigned* const fl0 = (const unsigned*)(myws + OFF_FLAGS + (size_t)(p * 8 * 40 + blk) * 128);
        const bool xskip = p == q && f;
        unsigned cc = lane < 16 ? 0u : 8u;
        unsigned left = 8;
        while (left) {
            bool ok = xskip;
            if (cc < 8u && !dead && !xskip)
                ok = (int)(__hip_atomic_load(fl0 + (size_t)cc * 40 * 32, __ATOMIC_RELAXED, __HIP_MEMORY_SCOPE_SYSTEM) - call) >= 0;
            const unsigned long long m = dead ? ~0ull : __ballot(ok);
            const unsigned n0 = n, left0 = left;
#pragma unroll
            for (int g6 = 0; g6 < 8; ++g6) {
                const unsigned cg6 = __builtin_amdgcn_readlane(cc, 2 * g6);
                if (cg6 < 8u && ((m >> (2 * g6)) & 3ull) == 3ull) {
                    const unsigned ps = (unsigned)(rank + g6) & 7u;
                    if (ps == q) { if (pp == g6) cc = 8u; --left; }
                    else {
                        if (lane == 0) lds_st(ctl + C_ORDER + n * 4, (ps << 3) | cg6);
                        if (pp == g6) ++cc;
                        ++n;
                        if (cg6 == 7u) --left;
                        if ((n & 7u) == 0u) PT(48 + (n >> 3));
                    }
                }
            }
            if (n != n0 || left != left0) {
                LGKM0();
                if (lane == 0) lds_st(ctl + C_READY, left == 0u ? 56u : (n < 56u ? n : 55u));
            } else idle();
        }
        {
            const unsigned* const wf = (const unsigned*)(myws + OFF_WFLAG) + (q * 8 + ntp) * 32 + (lane & 31);
            unsigned h = 0;
            while (h < 16u) {
                const bool ok = dead || (int)(__hip_atomic_load(wf, __ATOMIC_RELAXED, __HIP_MEMORY_SCOPE_AGENT) - call) >= 0;
                const unsigned m = (unsigned)__ballot(ok);
                const unsigned h0 = h;
                while (h < 16u && ((m >> (2 * h)) & 3u) == 3u) ++h;
                if (h != h0) { if (lane == 0) lds_st(ctl + C_READY, 56u + h); }
                else { __builtin_amdgcn_s_sleep(4); if ((++sp & 255u) == 0u) idle(); }
            }
        }
        PT_FLUSH(48, 56);
        return;
    }
    {
        PT_DECL;
        PT(0);
        __builtin_amdgcn_s_setprio(3);
        char* const dws = P.ws[q];
        unsigned* const dflags = (unsigned*)(dws + OFF_FLAGS + (size_t)rank * 8 * 40 * 128);
        unsigned* const bcnt = (unsigned*)(myws + OFF_BCNT + (size_t)q * 8 * 40 * 64);
        unsigned* const tick = (unsigned*)(myws + OFF_TICK + q * 128);
        const int g = lane >> 4, cc = lane & 15;
        const char* const xl = P.x + (size_t)cc * (KSH * 2) + g * 16;
        const char* const wl = P.w + (size_t)g * 8 * (NTOT * 2) + (size_t)q * (NSH * 2) + cc * 16;

        u32x4 R[8], T[8];
        auto mtile_of = [&](int type, int j) -> int {
            const int o = OROT ? (rank + 1 + (j >> 2)) & 7 : (j >> 2) + ((j >> 2) >= rank ? 1 : 0);
            return o * 4 + (type - 1) * 2 + ((j >> 1) & 1);
        };
        auto src_of = [&](unsigned u, const char*& base, unsigned& step_a, unsigned& step_b) {
            int type, c, j; unit_of(u, type, c, j);
            if (type == 0) {
                const int kh = (j >> 1) & 1, cg = (j >> 2) * 2 + (j & 1);
                base = wl + (size_t)(c * 64 + kh * 32) * (NTOT * 2) + cg * 256;
                step_a = NTOT * 2 * 2; step_b = NTOT * 2;
            } else {
                const int i = mtile_of(type, j), h = j & 1;
                base = xl + (size_t)(i * 128 + h * 64) * (KSH * 2) + c * 128;
                step_a = 16 * KSH * 2; step_b = 64;
            }
        };
        auto dst_of = [&](unsigned u, unsigned& off, unsigned& blk, int& type) {
            int c, j; unit_of(u, type, c, j);
            const unsigned ks = (unsigned)rank * 8 + c;
            if (type == 0) {
                const int kh = (j >> 1) & 1, cg = (j >> 2) * 2 + (j & 1);
                off = STAGE_W + (ks * 8 + (cg >> 1)) * 32768u + (kh * 2 + (cg & 1)) * 8192u;
                blk = c * 40 + ((j >> 2) >> WSH);
            } else {
                const int i = mtile_of(type, j), h = j & 1;
                off = (ks * 32 + i) * 16384u + h * 8192u;
                blk = c * 40 + 8 + (i >> XSH);
            }
        };
        unsigned zero_v;
        asm volatile("v_mov_b32 %0, 0" : "=v"(zero_v));
        auto draw = [&]() -> unsigned {
            unsigned t = 0;
            if (lane == 0) t = __hip_atomic_fetch_add(tick + zero_v, 1u, __ATOMIC_RELAXED, __HIP_MEMORY_SCOPE_AGENT);
            return t;
        };
#define LD(t) R[t] = *(const u32x4*)(nb + (size_t)((t) >> 1) * sa + (size_t)((t) & 1) * sbb)
#define ST(t) __builtin_amdgcn_raw_buffer_store_b128(T[t], srs, off + lane * 16 + (t) * 1024, 0, 17)
        unsigned ucur = (unsigned)idx;
        unsigned tnext = draw();
        {
            const char* nb; unsigned sa, sbb;
            src_of(ucur, nb, sa, sbb);
            LD(0); LD(1); LD(2); LD(3); LD(4); LD(5); LD(6); LD(7);
            __builtin_amdgcn_s_barrier();
#pragma unroll
            for (int t = 0; t < 4; ++t) {
                asm volatile("" ::: "memory");
                __hip_atomic_store(ctr + 2 + zero_v + t, 0u, __ATOMIC_RELAXED, __HIP_MEMORY_SCOPE_AGENT);
            }
        }
        asm volatile("" ::: "memory");
        const unsigned call = __builtin_amdgcn_readfirstlane(ctrv) + 1u;
        const unsigned par = call & 1u;
        const auto srs = __builtin_amdgcn_make_buffer_rsrc(dws + OFF_STAGE + (size_t)par * STAGE_BYTES, 0, STAGE_BYTES, 0x00020000);
        const unsigned tbase = (call - 1u) * TICKS_PER_CALL - 32u;
        PT(1);
        unsigned prev_blk = ~0u, prev_mask = 0;
        for (;;) {
            SWAIT(0);
            const unsigned unext = __builtin_amdgcn_readfirstlane(tnext) - tbase;
            int type; unsigned off, blk; dst_of(ucur, off, blk, type);
            if (type == 0) {
                TRANSPOSE8(T, R);
            } else {
#pragma unroll
                for (int jj = 0; jj < 8; ++jj) T[jj] = R[jj];
            }
            const char* nb; unsigned sa, sbb;
            src_of(unext < NUNITS ? unext : ucur, nb, sa, sbb);
            unsigned cres = 0;
#define COUNT() do { if (lane == 0) cres = __hip_atomic_fetch_add((prev_blk != ~0u ? bcnt + (size_t)prev_blk * 16 : ctr + 8) + zero_v, 1u, __ATOMIC_RELAXED, __HIP_MEMORY_SCOPE_AGENT); } while (0)
#define FLAG() do { asm volatile("" : "+v"(cres)); if (lane == 0) { if (prev_blk != ~0u && (cres & prev_mask) == prev_mask) __hip_atomic_store(dflags + (size_t)prev_blk * 32, call, __ATOMIC_RELAXED, __HIP_MEMORY_SCOPE_SYSTEM); \
                                     else __hip_atomic_store(ctr + 1, call, __ATOMIC_RELAXED, __HIP_MEMORY_SCOPE_AGENT); } } while (0)
            if (FLAGPOS == 0) COUNT();
            ST(0); tnext = draw(); LD(0); LD(1);
            SWAIT(1); if (FLAGPOS == 1) COUNT(); if (CPOS == 1) FLAG(); ST(1); LD(2); LD(3);
            SWAIT(2); if (FLAGPOS == 2) COUNT(); if (CPOS == 2) FLAG(); ST(2); LD(4); LD(5);
            SWAIT(3); if (CPOS == 3) FLAG(); ST(3); LD(6); LD(7);
            SWAIT(4); ST(4);
            SWAIT(5); ST(5);
            SWAIT(6); ST(6);
            SWAIT(7); ST(7);
#undef FLAG
#undef COUNT
            prev_blk = blk;
            prev_mask = type == 0 ? (4u << WSH) - 1u : (2u << XSH) - 1u;
            if (unext >= NUNITS) break;
            ucur = unext;
        }
#undef LD
#undef ST
        VMCNT(0);
        if (lane == 0) {
            const unsigned cres = __hip_atomic_fetch_add(bcnt + (size_t)prev_blk * 16 + zero_v, 1u, __ATOMIC_RELAXED, __HIP_MEMORY_SCOPE_AGENT);
            if ((cres & prev_mask) == prev_mask) __hip_atomic_store(dflags + (size_t)prev_blk * 32, call, __ATOMIC_RELAXED, __HIP_MEMORY_SCOPE_SYSTEM);
        }
        PT(26);
        lds_wait_all(ctl, C_YD, 8, 1u, lane);
        PT(27);
        const bool bad = __builtin_amdgcn_readfirstlane(__hip_atomic_load(tmo, __ATOMIC_RELAXED, __HIP_MEMORY_SCOPE_AGENT)) == call
                         || __builtin_amdgcn_readfirstlane(lds_ld(ctl + C_ABORT)) != 0u;
        {
            const auto ors = __builtin_amdgcn_make_buffer_rsrc(P.ws[q] + OFF_YP, 0, YP_BYTES, 0x00020000);
            const unsigned voff = (unsigned)(rank * 32 + ml * 8 + nt) * 65536u + lane * 16;
            const char* const yl = smem + LDS_YT + g * 2048 + cc * 16;
            unsigned* const act = (unsigned*)(myws + OFF_ACT + q * 128);
            unsigned left = 0;
#pragma unroll 1
            for (int rp = 0; rp < 8; ++rp) {
                const int gr = rp;
                u32x4 v[8];
#pragma unroll
                for (int k = 0; k < 8; ++k) {
                    v[k] = *(const u32x4*)(yl + (gr * 16 + (k >> 1)) * 512 + (k & 1) * 256);
                    if (bad) v[k] = (u32x4){0x7FC07FC0u, 0x7FC07FC0u, 0x7FC07FC0u, 0x7FC07FC0u};
                }
                unsigned w8 = YW0 * 8;
                if (rp > 0) {
                    const unsigned mine = 64u - 8u * (unsigned)rp;
                    unsigned tot = __builtin_amdgcn_readfirstlane(left) + (rp == 1 ? 56u : (unsigned)-8);
                    tot = tot < mine ? mine : (tot > 4096u ? 4096u : tot);
                    w8 = __builtin_amdgcn_readfirstlane((unsigned)((float)(YWT * 8) * (float)mine * __builtin_amdgcn_rcpf((float)tot)));
                    w8 = w8 < 8u ? 8u : (w8 > YWCAP * 8u ? YWCAP * 8u : w8);
                }
                {
                    const unsigned d = rp == 0 ? 56u : (unsigned)-8;
                    left = 0;
                    if (lane == 0) left = __hip_atomic_fetch_add(act + zero_v, d, __ATOMIC_RELAXED, __HIP_MEMORY_SCOPE_AGENT);
                }
#pragma unroll
                for (int k = 0; k < 8; ++k) {
                    const unsigned w = (w8 + (unsigned)(((k & 1) << 2) | (k & 2) | (k >> 2))) >> 3;
                    if (w <= 4u) {
                        if (w <= 2u) { if (w <= 1u) VMCNT(0); else VMCNT(1); }
                        else { if (w == 3u) VMCNT(2); else VMCNT(3); }
                    } else {
                        if (w <= 6u) { if (w == 5u) VMCNT(4); else VMCNT(5); }
                        else { if (w == 7u) VMCNT(6); else VMCNT(7); }
                    }
                    __builtin_amdgcn_raw_buffer_store_b128(v[k], ors, voff + (gr * 8 + k) * 1024, 0, 17);
                }
            }
            PT(28);
            PT(29);
            if (lane == 0) __hip_atomic_store(ctr, call, __ATOMIC_RELAXED, __HIP_MEMORY_SCOPE_AGENT);
        }
        PT(30);
        PT_FLUSH(0, 32);
    }
}

}

extern "C" void kernel_launch(int rank, int nranks, ncclComm_t comm,
                              hipStream_t stream,
                              void* const* d_in, const int* in_sizes, int n_in,
                              void* d_out, int out_size,
                              const int* in_dtypes, int out_dtype,
                              void* const* d_out_all,
                              void* const* d_ws_all, size_t ws_bytes) {
    (void)comm; (void)in_dtypes; (void)out_dtype; (void)d_out;
    if (nranks != NR || rank < 0 || rank >= NR || n_in < 2 || ws_bytes < WS_NEEDED
        || in_sizes[0] != MTOT * KSH || in_sizes[1] != KSH * NTOT || out_size != MSH * NTOT) {
        fprintf(stderr, "kernel_launch: unexpected problem geometry\n");
        return;
    }
    static bool first = true;
    if (first) {
        HIP_CHECK(hipFuncSetAttribute((const void*)fused_gemm_rs, hipFuncAttributeMaxDynamicSharedMemorySize, (int)LDS_TOTAL));
        first = false;
    }
    Params P;
    P.x = (const char*)d_in[0];
    P.w = (const char*)d_in[1];
    for (int p = 0; p < NR; ++p) { P.out[p] = (char*)d_out_all[p]; P.ws[p] = (char*)d_ws_all[p]; }
    P.rank = rank;
    fused_gemm_rs<<<NWG, NTHREADS, LDS_TOTAL, stream>>>(P);
    HIP_CHECK(hipGetLastError());
}
```
